```python
import numpy as np

kernel_name = "gpu_gemm_a2a_m4096_k4096_n4096_f32_reluqe4m3"


ABSMAX_RTOL = 0.09

COMPUTE_DTYPE = "float32"
EPILOGUE = "relu_quant_e4m3"
IN_QUANT = None
OUT_QUANT = "e4m3"
IN_DTYPES = {"x": "float32", "w_mat": "float32"}
WORLD_SIZE = 8

def setup_inputs():

    x = np.random.randn(4096, 4096).astype(np.float32)
    w_mat = (np.random.randn(4096, 4096) / np.sqrt(4096)).astype(np.float32)
    return {"x": x, "w_mat": w_mat}

def _snap_fp8(x, mant_bits, max_abs, min_normal):
    a = np.abs(x.astype(np.float64))
    _m, _e = np.frexp(a)
    q = np.where(a >= min_normal, np.ldexp(1.0, _e - 1 - mant_bits),
                 min_normal * 2.0 ** -mant_bits)
    snapped = np.minimum(np.round(a / q) * q, max_abs)
    return (np.sign(x) * snapped).astype(np.float32)
def _quant_dequant(y, amax):
    scale = amax / 448.0
    q = _snap_fp8(y / scale, 3, 448.0, 2.0 ** -6)
    return (q * scale).astype(np.float32)
def _epilogue(y):
    y = np.maximum(y, 0.0)
    return _quant_dequant(y, np.abs(y).max())

def reference(x, w_mat):
    y = (x @ w_mat).astype(np.float32)
    return _epilogue(y)
```

```cpp
#include <hip/hip_runtime.h>
#include <rccl/rccl.h>
#include <cstdio>

#define HIP_CHECK(expr) do { hipError_t _e = (expr); if (_e != hipSuccess) { \
    fprintf(stderr, "HIP error %s at %s:%d\n", hipGetErrorString(_e), __FILE__, __LINE__); \
    return; } } while (0)

typedef unsigned u32x4 __attribute__((ext_vector_type(4)));
typedef float f32x4 __attribute__((ext_vector_type(4)));
typedef float f32x2 __attribute__((ext_vector_type(2)));
typedef __bf16 bf16x2 __attribute__((ext_vector_type(2)));
typedef __bf16 bf16x8 __attribute__((ext_vector_type(8)));
typedef unsigned long long u64;

constexpr int NR = 8, M = 512, K = 4096, N = 4096, NS = N / NR;
constexpr int NWG = 256;
constexpr int RSRC_W3 = 0x00020000;
constexpr u64 DEADLINE = 50000000ull;
constexpr size_t WS_TMO = 0;
constexpr size_t WS_CNT = 4096;
constexpr size_t WS_AMAX = 65536;
constexpr size_t WS_DONE = 131072;
constexpr size_t WS_XB = 1u << 20;
constexpr size_t WS_NEED = WS_XB + (size_t)M * K * 2;

struct Params {
  const float* x; const float* w;
  unsigned char* out[NR];
  unsigned char* ws[NR];
  int rank; int phase;
};

__device__ __forceinline__ unsigned cvt2(unsigned a, unsigned b) {
  f32x2 v = {__builtin_bit_cast(float, a), __builtin_bit_cast(float, b)};
  bf16x2 r = __builtin_convertvector(v, bf16x2);
  return __builtin_bit_cast(unsigned, r);
}
__device__ __forceinline__ f32x4 mfma(u32x4 a, u32x4 b, f32x4 c) {
  return __builtin_amdgcn_mfma_f32_16x16x32_bf16(__builtin_bit_cast(bf16x8, a), __builtin_bit_cast(bf16x8, b), c, 0, 0, 0);
}
#define SB() __builtin_amdgcn_sched_barrier(0)
__device__ __forceinline__ void st16_sys(void* p, u32x4 v) {
  asm volatile("global_store_dwordx4 %0, %1, off sc0 sc1\n\ts_nop 1" :: "v"(p), "v"(v) : "memory");
}

__global__ void __launch_bounds__(256, 1) fused(Params p) {
  extern __shared__ __attribute__((aligned(16))) unsigned char smem[];
  const int tid = threadIdx.x, lane = tid & 63, wv = __builtin_amdgcn_readfirstlane(tid >> 6);
  const int b = blockIdx.x;
  unsigned char* ws = p.ws[p.rank];
  if (p.phase == 0) {
    u32x4* xb = (u32x4*)(ws + WS_XB);
    int t = b * 256 + tid;
    int ks = (t >> 6) & 127, mb = t >> 13;
    int r = lane & 15, g = lane >> 4;
    const u32x4* q = (const u32x4*)(p.x + (size_t)(mb * 16 + r) * K + ks * 32 + 8 * g);
    u32x4 a = q[0], c = q[1];
    xb[t] = u32x4{cvt2(a.x, a.y), cvt2(a.z, a.w), cvt2(c.x, c.y), cvt2(c.z, c.w)};
    return;
  }
  unsigned call = 0;
  if (tid == 0) call = __hip_atomic_fetch_add((unsigned*)(ws + WS_CNT + b * 128), 1u, __ATOMIC_RELAXED, __HIP_MEMORY_SCOPE_AGENT) + 1u;
  constexpr int NW = 4, D = 2;
  const int xcd = b & 7, idx = b >> 3;
  const int n0 = xcd * 512 + (idx & 7) * 64, m0 = (idx >> 3) * 128;
  const int r = lane & 15, g = lane >> 4;
  const __amdgpu_buffer_rsrc_t rw = __builtin_amdgcn_make_buffer_rsrc((void*)p.w, 0, K * N * 4, RSRC_W3);
  const __amdgpu_buffer_rsrc_t rx = __builtin_amdgcn_make_buffer_rsrc((void*)(ws + WS_XB), 0, M * K * 2, RSRC_W3);
  const int wvoff = 8 * g * (N * 4) + (n0 + 4 * r) * 4;
  const int xvoff = (m0 / 16) * (128 * 1024) + lane * 16;
  u32x4 wb[D][8], xf[D][8];
  f32x4 acc[8][4];
#pragma unroll
  for (int i = 0; i < 8; ++i)
#pragma unroll
    for (int c = 0; c < 4; ++c) acc[i][c] = f32x4{0.f, 0.f, 0.f, 0.f};
  constexpr int NKS = 128 / NW;
#define LDW(s, ks, j) wb[s][j] = __builtin_amdgcn_raw_buffer_load_b128(rw, wvoff, ((ks) * 32 + (j)) * (N * 4), 0)
#define LDX(s, ks, mb) xf[s][mb] = __builtin_amdgcn_raw_buffer_load_b128(rx, xvoff, (mb) * (128 * 1024) + (ks) * 1024, 0)
#pragma unroll
  for (int s = 0; s < D; ++s) {
    const int ks = wv + s * NW;
#pragma unroll
    for (int j = 0; j < 8; ++j) LDW(s, ks, j);
#pragma unroll
    for (int mb = 0; mb < 8; ++mb) LDX(s, ks, mb);
  }
  SB();
#define STEP(s, ksn, LOAD) do { \
      u32x4 af[4]; \
      af[0] = u32x4{cvt2(wb[s][0].x, wb[s][1].x), cvt2(wb[s][2].x, wb[s][3].x), cvt2(wb[s][4].x, wb[s][5].x), cvt2(wb[s][6].x, wb[s][7].x)}; \
      af[1] = u32x4{cvt2(wb[s][0].y, wb[s][1].y), cvt2(wb[s][2].y, wb[s][3].y), cvt2(wb[s][4].y, wb[s][5].y), cvt2(wb[s][6].y, wb[s][7].y)}; \
      af[2] = u32x4{cvt2(wb[s][0].z, wb[s][1].z), cvt2(wb[s][2].z, wb[s][3].z), cvt2(wb[s][4].z, wb[s][5].z), cvt2(wb[s][6].z, wb[s][7].z)}; \
      af[3] = u32x4{cvt2(wb[s][0].w, wb[s][1].w), cvt2(wb[s][2].w, wb[s][3].w), cvt2(wb[s][4].w, wb[s][5].w), cvt2(wb[s][6].w, wb[s][7].w)}; \
      SB(); \
      if (LOAD) { _Pragma("unroll") for (int j = 0; j < 8; ++j) LDW(s, ksn, j); } \
      SB(); \
      _Pragma("unroll") for (int mb = 0; mb < 8; ++mb) { \
        u32x4 bf = xf[s][mb]; \
        _Pragma("unroll") for (int c = 0; c < 4; ++c) acc[mb][c] = mfma(af[c], bf, acc[mb][c]); \
        if (LOAD) LDX(s, ksn, mb); \
        SB(); \
      } } while (0)
  for (int it = 0; it < NKS - D; it += D) {
#pragma unroll
    for (int s = 0; s < D; ++s) { const int ksn = wv + (it + s + D) * NW; STEP(s, ksn, true); }
  }
#pragma unroll
  for (int s = 0; s < D; ++s) STEP(s, 0, false);

  f32x4* red = (f32x4*)smem;
  float* sm_max = (float*)(smem + 131072);
  unsigned* sm_call = (unsigned*)(smem + 131072 + 16);
  float* sm_gmax = (float*)(smem + 131072 + 32);
#pragma unroll
  for (int mb = 0; mb < 8; ++mb)
#pragma unroll
    for (int c = 0; c < 4; ++c) red[((wv * 8 + mb) * 4 + c) * 64 + lane] = acc[mb][c];
  if (tid == 0) *sm_call = call;
  __syncthreads();
  call = *sm_call;
  float amax = 0.f;
  f32x4 fin[2][4];
#pragma unroll
  for (int i = 0; i < 2; ++i) {
    const int mb = wv * 2 + i;
#pragma unroll
    for (int c = 0; c < 4; ++c) {
      f32x4 s = {0.f, 0.f, 0.f, 0.f};
#pragma unroll
      for (int ww = 0; ww < 4; ++ww) s += red[((ww * 8 + mb) * 4 + c) * 64 + lane];
      fin[i][c] = s;
      amax = fmaxf(amax, fmaxf(fmaxf(s.x, s.y), fmaxf(s.z, s.w)));
    }
  }
#pragma unroll
  for (int o = 32; o >= 1; o >>= 1) amax = fmaxf(amax, __shfl_xor(amax, o));
  if (lane == 0) sm_max[wv] = amax;
  __syncthreads();
  unsigned* tmo = (unsigned*)(ws + WS_TMO);
  const size_t goff = WS_AMAX + (size_t)(call & 1) * (NR * NWG * 8);
  if (tid < NR) {
    float m = fmaxf(fmaxf(sm_max[0], sm_max[1]), fmaxf(sm_max[2], sm_max[3]));
    u64 gr = ((u64)call << 32) | __builtin_bit_cast(unsigned, m);
    __hip_atomic_store((u64*)(p.ws[tid] + goff + (p.rank * NWG + b) * 8), gr, __ATOMIC_RELAXED, __HIP_MEMORY_SCOPE_SYSTEM);
  }
  {
    const u64* gp = (const u64*)(ws + goff) + tid;
    float gm = 0.f;
    const u64 t0 = __builtin_amdgcn_s_memrealtime();
    for (;;) {
      bool ok = true; gm = 0.f;
#pragma unroll
      for (int i = 0; i < 8; ++i) {
        u64 v = __hip_atomic_load(gp + i * 256, __ATOMIC_RELAXED, __HIP_MEMORY_SCOPE_SYSTEM);
        ok &= (unsigned)(v >> 32) == call;
        gm = fmaxf(gm, __builtin_bit_cast(float, (unsigned)v));
      }
      if (__all(ok)) break;
      if (__hip_atomic_load(tmo, __ATOMIC_RELAXED, __HIP_MEMORY_SCOPE_AGENT) == call || __builtin_amdgcn_s_memrealtime() - t0 > DEADLINE) {
        __hip_atomic_store(tmo, call, __ATOMIC_RELAXED, __HIP_MEMORY_SCOPE_AGENT);
        break;
      }
      __builtin_amdgcn_s_sleep(2);
    }
#pragma unroll
    for (int o = 32; o >= 1; o >>= 1) gm = fmaxf(gm, __shfl_xor(gm, o));
    if (lane == 0) sm_gmax[wv] = gm;
  }
  __syncthreads();
  const float gmax = fmaxf(fmaxf(sm_gmax[0], sm_gmax[1]), fmaxf(sm_gmax[2], sm_gmax[3]));
  const bool bad = __hip_atomic_load(tmo, __ATOMIC_RELAXED, __HIP_MEMORY_SCOPE_AGENT) == call;
  const float inv_scale = gmax > 0.f ? 448.f / gmax : 0.f;
  unsigned char* dst = p.out[xcd];
#pragma unroll
  for (int i = 0; i < 2; ++i) {
    const int mb = wv * 2 + i;
    u32x4 o;
#pragma unroll
    for (int q = 0; q < 4; ++q) {
      float v0 = __builtin_amdgcn_fmed3f(fin[i][0][q] * inv_scale, 0.f, 448.f), v1 = __builtin_amdgcn_fmed3f(fin[i][1][q] * inv_scale, 0.f, 448.f);
      float v2 = __builtin_amdgcn_fmed3f(fin[i][2][q] * inv_scale, 0.f, 448.f), v3 = __builtin_amdgcn_fmed3f(fin[i][3][q] * inv_scale, 0.f, 448.f);
      int pk = __builtin_amdgcn_cvt_pk_fp8_f32(v0, v1, 0, false);
      pk = __builtin_amdgcn_cvt_pk_fp8_f32(v2, v3, pk, true);
      o[q] = bad ? 0x7f7f7f7fu : (unsigned)pk;
    }
    st16_sys(dst + (size_t)(p.rank * M + m0 + mb * 16 + r) * NS + (n0 - xcd * NS) + 16 * g, o);
  }
  asm volatile("s_waitcnt vmcnt(0)" ::: "memory");
  __syncthreads();
  if (tid == 0) {
    __hip_atomic_store((unsigned*)(p.ws[xcd] + WS_DONE) + p.rank * 32 + idx, call, __ATOMIC_RELAXED, __HIP_MEMORY_SCOPE_SYSTEM);
    const unsigned* f = (const unsigned*)(ws + WS_DONE) + b;
    const u64 t0 = __builtin_amdgcn_s_memrealtime();
    while (__hip_atomic_load(f, __ATOMIC_RELAXED, __HIP_MEMORY_SCOPE_SYSTEM) < call) {
      if (__hip_atomic_load(tmo, __ATOMIC_RELAXED, __HIP_MEMORY_SCOPE_AGENT) == call || __builtin_amdgcn_s_memrealtime() - t0 > DEADLINE) {
        __hip_atomic_store(tmo, call, __ATOMIC_RELAXED, __HIP_MEMORY_SCOPE_AGENT);
        __hip_atomic_store(p.out[p.rank] + (size_t)p.rank * M * NS, (unsigned char)0x7f, __ATOMIC_RELAXED, __HIP_MEMORY_SCOPE_SYSTEM);
        break;
      }
      __builtin_amdgcn_s_sleep(2);
    }
  }
}

extern "C" void kernel_launch(int rank, int nranks, ncclComm_t comm, hipStream_t stream,
                              void* const* d_in, const int* in_sizes, int n_in,
                              void* d_out, int out_size, const int* in_dtypes, int out_dtype,
                              void* const* d_out_all, void* const* d_ws_all, size_t ws_bytes) {
  if (nranks != NR || (unsigned)rank >= (unsigned)NR || n_in < 2 || in_sizes[0] != M * K || in_sizes[1] != K * N || out_size != NR * M * NS || ws_bytes < WS_NEED) {
    fprintf(stderr, "unexpected problem shape\n");
    return;
  }
  constexpr int LDS = 131072 + 64;
  static bool init = false;
  if (!init) { HIP_CHECK(hipFuncSetAttribute((const void*)fused, hipFuncAttributeMaxDynamicSharedMemorySize, LDS)); init = true; }
  Params p;
  p.x = (const float*)d_in[0]; p.w = (const float*)d_in[1];
  for (int i = 0; i < NR; ++i) { p.out[i] = (unsigned char*)d_out_all[i]; p.ws[i] = (unsigned char*)d_ws_all[i]; }
  p.rank = rank;
  p.phase = 0;
  hipLaunchKernelGGL(fused, dim3(32 * 128 * 64 / 256), dim3(256), 0, stream, p);
  p.phase = 1;
  hipLaunchKernelGGL(fused, dim3(NWG), dim3(256), LDS, stream, p);
  HIP_CHECK(hipGetLastError());
}
```

```cpp
#include <hip/hip_runtime.h>
#include <rccl/rccl.h>
#include <cstdio>
#include <type_traits>
#include <utility>

typedef unsigned u32x4 __attribute__((ext_vector_type(4)));
typedef float f32x4 __attribute__((ext_vector_type(4)));
typedef float f32x2 __attribute__((ext_vector_type(2)));
typedef __bf16 bf16x2 __attribute__((ext_vector_type(2)));
typedef unsigned long long u64;
#define LAS __attribute__((address_space(3)))
#define GAS __attribute__((address_space(1)))
template <class T> __device__ __forceinline__ GAS T* G(T* p) { return (GAS T*)p; }

#define ROT (nt + 8 * nh)
#define QUIET 48
#define RET0 22
#define EA 4
#define W0A 5
#define PD 0
#define ZMF 1
#define SSP 5
#define FL1 104
#define FL2 134
#define CLR 60
#define PLAIN 0
#define RUP 64
#define PDLY 76
#define PCLS (wv * PDLY)
#define MSL 2
#define XNT 0
#define CPF0 1

constexpr int NR = 8, M = 512, K = 4096, N = 4096, NS = N / NR;
constexpr int NWG = 256;
constexpr int RSRC_W3 = 0x00020000;
constexpr int LOC = 16;
constexpr u64 DEADLINE = 50000000ull;
constexpr size_t WS_TMO = 0;
constexpr size_t WS_GMAX = 1024;
constexpr size_t WS_CNT = 4096;
constexpr size_t WS_AMAX = 65536;
constexpr size_t WS_CFLAG = 131072;
constexpr size_t WS_XFLAG = 524288;
constexpr size_t WS_XB = 1u << 20;
constexpr size_t WS_SLAB = 8u << 20;
constexpr size_t WS_STG = 24u << 20;
constexpr size_t WS_NEED = WS_STG + (8u << 20);
constexpr int LDS_MISC = 6 * 16384;
constexpr int LDS_BYTES = LDS_MISC + 256;
constexpr unsigned BADMAX = 0x7f800000u;

struct Params {
  const float* x; const float* w;
  unsigned char* wsl;
  unsigned char* outl;
  int rank; int pad;
  unsigned char* out[NR];
  unsigned char* ws[NR];
};

__device__ __forceinline__ float asf(unsigned a) { return __builtin_bit_cast(float, a); }
__device__ __forceinline__ unsigned asu(float a) { return __builtin_bit_cast(unsigned, a); }
__device__ __forceinline__ unsigned cvt2f(float a, float b) {
  f32x2 v = {a, b};
  return __builtin_bit_cast(unsigned, __builtin_convertvector(v, bf16x2));
}
__device__ __forceinline__ unsigned cvt2(unsigned a, unsigned b) { return cvt2f(asf(a), asf(b)); }
#define SB() __builtin_amdgcn_sched_barrier(0)
#define RFL(x) __builtin_amdgcn_readfirstlane(x)
#define BAR() do { __builtin_amdgcn_s_waitcnt(0xC07F); asm volatile("" ::: "memory"); __builtin_amdgcn_s_barrier(); asm volatile("" ::: "memory"); } while (0)
#define DRAIN() asm volatile("s_waitcnt vmcnt(0)" ::: "memory")
#define ORDER() __builtin_amdgcn_fence(__ATOMIC_ACQUIRE, "wavefront")
__device__ __forceinline__ void st16_sys(void* p, u32x4 v) {
  asm volatile("global_store_dwordx4 %0, %1, off sc0 sc1\n\ts_nop 1" :: "v"(p), "v"(v) : "memory");
}
__device__ __forceinline__ void st16_loc(void* p, u32x4 v) {
  asm volatile("global_store_dwordx4 %0, %1, off sc1\n\ts_nop 1" :: "v"(p), "v"(v) : "memory");
}
#define FLINE(pol) asm volatile("s_mov_b64 exec, 0xff\n\tglobal_store_dwordx4 %0, %1, off " pol "\n\ts_mov_b64 exec, -1\n\ts_nop 0" \
                                :: "v"((unsigned char*)p + (lane & 7) * 16), "v"(u32x4{v, v, v, v}) : "memory")
__device__ __forceinline__ void flag_loc(void* p, unsigned v, int lane) { FLINE("sc1"); }
__device__ __forceinline__ void flag_sys(void* p, unsigned v, int lane) { FLINE("sc0 sc1"); }
__device__ __forceinline__ unsigned ld_loc(const void* p) { return __hip_atomic_load(G((const unsigned*)p), __ATOMIC_RELAXED, __HIP_MEMORY_SCOPE_AGENT); }
template <int CTRL, int ROWMASK> __device__ __forceinline__ unsigned dpp_max(unsigned v) {
  return max(v, (unsigned)__builtin_amdgcn_update_dpp(0, (int)v, CTRL, ROWMASK, 0xf, false));
}
__device__ __forceinline__ unsigned wave_max(unsigned v) {
  v = dpp_max<0x111, 0xf>(v); v = dpp_max<0x112, 0xf>(v); v = dpp_max<0x114, 0xf>(v); v = dpp_max<0x118, 0xf>(v);
  v = dpp_max<0x142, 0xa>(v); v = dpp_max<0x143, 0xc>(v);
  return (unsigned)__builtin_amdgcn_readlane((int)v, 63);
}
template <int CTRL, int ROWMASK> __device__ __forceinline__ unsigned dpp_add(unsigned v) {
  return v + (unsigned)__builtin_amdgcn_update_dpp(0, (int)v, CTRL, ROWMASK, 0xf, false);
}
__device__ __forceinline__ unsigned wave_scan(unsigned v) {
  v = dpp_add<0x111, 0xf>(v); v = dpp_add<0x112, 0xf>(v); v = dpp_add<0x114, 0xf>(v); v = dpp_add<0x118, 0xf>(v);
  v = dpp_add<0x142, 0xa>(v); v = dpp_add<0x143, 0xc>(v);
  return v;
}
__device__ __forceinline__ u32x4 quant16(const f32x4 (&d)[4], float inv) {
  u32x4 o;
#pragma unroll
  for (int q = 0; q < 4; ++q) {
    int pk = __builtin_amdgcn_cvt_pk_fp8_f32(fminf(d[q].x * inv, 448.f), fminf(d[q].y * inv, 448.f), 0, false);
    o[q] = (unsigned)__builtin_amdgcn_cvt_pk_fp8_f32(fminf(d[q].z * inv, 448.f), fminf(d[q].w * inv, 448.f), pk, true);
  }
  return o;
}
template <int I> using IC = std::integral_constant<int, I>;
template <class F, int... Is> __device__ __forceinline__ void sfor_impl(F&& f, std::integer_sequence<int, Is...>) { (f(IC<Is>{}), ...); }
template <int NN, class F> __device__ __forceinline__ void sfor(F&& f) { sfor_impl(f, std::make_integer_sequence<int, NN>{}); }
constexpr int late_load(int i) { for (int k = 0; k < 18; ++k) if (i == QUIET + (k * (96 - QUIET)) / 18) return k; return -1; }
struct Waiter {
  unsigned* tmo; unsigned call; u64 t0; unsigned n;
  __device__ __forceinline__ Waiter(unsigned* t, unsigned c) : tmo(t), call(c), t0(__builtin_amdgcn_s_memrealtime()), n(0) {}
  __device__ __forceinline__ bool giveup() {
    if ((++n & 63u) == 0u) {
      if (RFL(ld_loc(tmo)) == call || __builtin_amdgcn_s_memrealtime() - t0 > DEADLINE) {
        __hip_atomic_store(G(tmo), call, __ATOMIC_RELAXED, __HIP_MEMORY_SCOPE_AGENT);
        return true;
      }
    }
    return false;
  }
};
#define CODE_PF(dst, beg, end) do { u64 pc_; unsigned len_; \
    asm volatile("s_getpc_b64 %0\n" beg ":\n\ts_mov_b32 %1, " end "-" beg : "=s"(pc_), "=s"(len_)); \
    dst = __builtin_amdgcn_raw_buffer_load_b32(__builtin_amdgcn_make_buffer_rsrc((void*)pc_, 0, len_, RSRC_W3), (lane * 4 + wv) * 64, 0, 0); SB(); } while (0)

template <int MT> __device__ __forceinline__ void body(const Params& p, u32x4 (&t)[8], unsigned call) {
  extern __shared__ __attribute__((aligned(16))) unsigned char smem[];
  const int tid = threadIdx.x, lane = tid & 63, wv = RFL(tid >> 6);
  const int b = blockIdx.x;
  unsigned char* ws = (unsigned char*)(((u64)RFL((unsigned)((u64)p.wsl >> 32)) << 32) | (unsigned)RFL((unsigned)(u64)p.wsl));
  unsigned cpf0;
  if constexpr (MT == 0) CODE_PF(cpf0, ".Lz_a0", ".Lz_b0"); else CODE_PF(cpf0, ".Lz_a1", ".Lz_b1");
  const int kq = b & 3, nh = (b >> 2) & 1, nt = b >> 4, ntg = (nt >> 1) * 4 + nh * 2 + (nt & 1);
  const int r = lane & 15, g = lane >> 4;
  const int dest = ntg >> 2;
  unsigned char* out_dest = p.out[dest];
  unsigned char* ws_dest = p.ws[dest];
  unsigned char* out_own = p.outl;
  int rank = p.rank;
  constexpr int nsync = NR;
  u64 wsp[NR];
#pragma unroll
  for (int i = 0; i < NR; ++i) wsp[i] = (u64)p.ws[i];
  LAS unsigned char* L = (LAS unsigned char*)smem;
  unsigned* tmo = (unsigned*)(ws + WS_TMO);
  const __amdgpu_buffer_rsrc_t rw = __builtin_amdgcn_make_buffer_rsrc((void*)p.w, 0, K * N * 4, RSRC_W3);
  const __amdgpu_buffer_rsrc_t rx = __builtin_amdgcn_make_buffer_rsrc((void*)(ws + WS_XB), 0, M * K * 2, RSRC_W3);
  const int wvoff = (kq * 1024 + 16 * (wv >> 1) + 4 * g) * (N * 4) + (ntg * 128 + (wv & 1) * 64 + 4 * r) * 4;
  const int xvoff = (kq * 32) * 32768 + (MT * 16 + wv * 4) * 1024 + lane * 16;
  const int wwoff = wv * 4096 + lane * 16;
  int mbo[4];
#pragma unroll
  for (int j = 0; j < 4; ++j) mbo[j] = ((j + kq) & 3) * 1024;
  const int rot16 = (ROT) & 15;
#define CH(s) (((s) + rot16) & 15)
#define LDWJ(dst, c, j) dst[j] = __builtin_amdgcn_raw_buffer_load_b128(rw, wvoff, (c) * (64 * N * 4) + (((j) & 3) + 32 * ((j) >> 2)) * (N * 4), 0)
#define LDXK(dst, tk, j) dst[j] = __builtin_amdgcn_raw_buffer_load_b128(rx, xvoff, mbo[j] + (CH((tk) >> 1) * 2 + ((tk) & 1)) * 32768, LOC)
#define CV(src, e, q) tf[q] = cvt2(src[2 * (q)][e], src[2 * (q) + 1][e])
#define RET(src, off, n) do { if ((n) % 5 < 4) CV(src, (n) / 5, (n) % 5 & 3); else *(LAS u32x4*)(L + (off) + ((n) / 5) * 1024) = tf; } while (0)
#define RETALL(src, off) do { _Pragma("unroll") for (int n_ = 0; n_ < 20; ++n_) RET(src, off, n_); } while (0)
#define CVH(n) H[n] = cvt2(U[1][2 * ((n) & 1)][(n) >> 1], U[1][2 * ((n) & 1) + 1][(n) >> 1])
#define RETH(off, n) do { if ((n) % 3 < 2) CV(U[1], (n) / 3, 2 + ((n) % 3 & 1)); \
    else *(LAS u32x4*)(L + (off) + ((n) / 3) * 1024) = u32x4{H[2 * ((n) / 3)], H[2 * ((n) / 3) + 1], tf[2], tf[3]}; } while (0)
#define RDF(h, off, f) fw[h][f] = *(const LAS u32x4*)(L + (off) + (f) * 1024)
#define MF(j, f, h, xb) asm volatile("v_mfma_f32_16x16x32_bf16 %0, %1, %2, %0" : "+a"(acc[j][f]) : "v"(fw[h][f]), "a"(xb))
#define A8(j) "+a"(acc[j][0]), "+a"(acc[j][1]), "+a"(acc[j][2]), "+a"(acc[j][3]), "+a"(acc[j][4]), "+a"(acc[j][5]), "+a"(acc[j][6]), "+a"(acc[j][7])
  u32x4 X[6][4], U[3][8], fw[2][8], tf;
  unsigned H[8];
  f32x4 acc[4][8];
  bool bad = false;
  unsigned wl_lo = 0, wl_hi = 0;
  unsigned fall;
  const unsigned char* fp = ws + WS_CFLAG + (size_t)(((kq * 2 + MT) * 8 + wv * 2) * 16 + (lane & 31)) * 128;
  {
    u32x4 wa[8], wb[8];
    const int c0 = CH(MT), c1 = CH(1 - MT), c2 = CH(2), c3 = CH(3);
#define PL(n) do { if ((n) < 8) LDWJ(wa, c0, (n) & 7); else LDWJ(U[1], c2, 2 * MT + ((n) & 1)); } while (0)
#pragma unroll
    for (int n = 0; n < EA; ++n) PL(n);
    SB();
    u32x4 z4 = {0u, 0u, 0u, 0u};
    asm volatile("s_nop 1" : "+v"(z4));
#pragma unroll
    for (int j = 0; j < 4; ++j)
#pragma unroll
      for (int f = 0; f < 8; ++f) asm volatile("v_mfma_f32_16x16x32_bf16 %0, %1, %1, 0" : "=a"(acc[j][f]) : "v"(z4));
    SB();
    const int row0 = MT * 256 + nh * 128 + wv * 32, pc = lane & 7;
    unsigned char* d = ws + WS_XB + (size_t)((kq * 32 + nt * 2 + (pc >> 2)) * 32 + (row0 >> 4)) * 1024 + ((pc & 3) * 16 + (lane >> 3)) * 16;
    u32x4 o[4];
    asm volatile("" : "+v"(t[0]), "+v"(t[1]), "+v"(t[2]), "+v"(t[3]), "+v"(t[4]), "+v"(t[5]), "+v"(t[6]), "+v"(t[7]), "+v"(call));
#pragma unroll
    for (int i = 0; i < 4; ++i) o[i] = u32x4{cvt2(t[2 * i].x, t[2 * i].y), cvt2(t[2 * i].z, t[2 * i].w), cvt2(t[2 * i + 1].x, t[2 * i + 1].y), cvt2(t[2 * i + 1].z, t[2 * i + 1].w)};
    SB();
#pragma unroll
    for (int i = 0; i < 4; ++i) st16_loc(d + (i >> 1) * 1024 + (i & 1) * 128, o[i]);
    call = RFL(call) + 1u;
    if (tid == 0) *(LAS unsigned*)(L + LDS_MISC + 20) = 0u;
    SB();
    __builtin_amdgcn_s_barrier();
    SB();
#pragma unroll
    for (int n = EA; n < EA + W0A; ++n) PL(n);
    SB();
    asm volatile("s_waitcnt vmcnt(%0)" :: "n"(W0A) : "memory");
    flag_loc(ws + WS_CFLAG + (size_t)(((kq * 2 + MT) * 8 + nh * 4 + wv) * 16 + nt) * 128, call, lane);
    SB();
#pragma unroll
    for (int n = EA + W0A; n < 10; ++n) PL(n);
    SB();
#pragma unroll
    for (int i = 0; i < NR; ++i)
      asm volatile("v_writelane_b32 %0, %2, %4\n\tv_writelane_b32 %1, %3, %4" : "+v"(wl_lo), "+v"(wl_hi) : "s"((unsigned)wsp[i]), "s"((unsigned)(wsp[i] >> 32)), "n"(i));
    asm volatile("" : "+s"(out_own), "+s"(out_dest), "+s"(ws_dest), "+s"(rank));
    RETALL(wa, MT * 16384 + wwoff);
    asm volatile("" :: "v"(cpf0));
    SB();
    const unsigned char* f2 = ws + WS_CFLAG + (size_t)(((kq * 2 + MT) * 8 + wv * 2 + (lane & 1)) * 16 + CH((lane >> 1) & 1)) * 128;
    unsigned f0 = ld_loc(f2);
    SB();
#pragma unroll
    for (int j = 0; j < 8; ++j) LDWJ(wb, c1, j);
    LDWJ(U[1], c2, 2 - 2 * MT); LDWJ(U[1], c2, 3 - 2 * MT);
    SB();
    Waiter wt(tmo, call);
    if (!__all(f0 >= call)) {
      while (!__all(lane >= 4 || ld_loc(f2) >= call)) {
        if (wt.giveup()) { bad = true; break; }
      }
    }
    ORDER();
#pragma unroll
    for (int q = 0; q < 12; ++q) LDXK(X[q >> 2], q >> 2, q & 3);
    SB();
    RETALL(wb, (1 - MT) * 16384 + wwoff);
#pragma unroll
    for (int n = 0; n < 8; ++n) CVH(n);
    SB();
#pragma unroll
    for (int q = 0; q < 12; ++q) { const int n = 12 + (q + 6 * MT) % 12; if (n < 16) LDWJ(U[1], c2, n & 7); else LDWJ(U[2], c3, n & 7); }
    SB();
    fall = ld_loc(fp);
    SB();
  }
  BAR();
#pragma unroll
  for (int f = 0; f < 8; ++f) RDF(0, lane * 16, f);
  SB();
  {
    int rcur = lane * 16, roth = lane * 16 + 49152, wcur = wwoff, woth = wwoff + 49152;
    for (int it = 0; it < 4; ++it) {
      const int cw[3] = {CH(3 * it + 4), CH(3 * it + 5), CH(3 * it + 6)};
      sfor<192>([&](auto ic) __attribute__((always_inline)) {
        constexpr int P = decltype(ic)::value / 96, i = decltype(ic)::value % 96, tt = 3 * P + i / 32, ii = i % 32, k = late_load(i);
        MF(ii >> 3, ii & 7, tt & 1, X[tt][ii >> 3]);
        if constexpr (i < 24 && i % 4 == 0) { constexpr int n = 12 * P + 6 * MT + i / 4; LDWJ(U[n >> 3], cw[n >> 3], n & 7); }
        if constexpr (k >= 12) { constexpr int n = 12 * P + 6 * (1 - MT) + k - 12; LDWJ(U[n >> 3], cw[n >> 3], n & 7); }
        if constexpr (k >= 0 && k < 12) { constexpr int t2 = 3 * P + 3 + k / 4; LDXK(X[t2 % 6], 6 * it + t2, k & 3); }
        if constexpr (ii < 24 && ii % 3 == 1) { if constexpr (tt < 5) RDF((tt + 1) & 1, rcur + (tt + 1) * 8192, ii / 3); else RDF(0, roth, ii / 3); }
        if constexpr (P == 0 && i >= RET0 && i < RET0 + 12) RETH(wcur + 32768, i - RET0);
        if constexpr (P == 0 && i >= RET0 + 12 && i < RET0 + 32) RET(U[2], woth, i - RET0 - 12);
        if constexpr (P == 1 && i >= RET0 && i < RET0 + 20) RET(U[0], woth + 16384, i - RET0);
        if constexpr (P == 1 && i >= RET0 + 20 && i < RET0 + 28) CVH(i - RET0 - 20);
        SB();
        if constexpr (P == 0 && i == QUIET - 1) {
          if (it == 0) {
            asm volatile("" : "+v"(fall));
            if (!__all(fall >= call) && !bad) {
              Waiter wt(tmo, call);
              while (!__all(lane >= 32 || ld_loc(fp) >= call)) {
                if (wt.giveup()) { bad = true; break; }
              }
            }
            ORDER();
          }
        }
        if constexpr (i == 95) { BAR(); }
      });
      const int sw = rcur; rcur = roth; roth = sw;
      const int sw2 = wcur; wcur = woth; woth = sw2;
    }
  }
#define XF(j) (ws + WS_XFLAG + (size_t)((((b & ~3) | ((kq + (j)) & 3)) * 4 + wv) * 4 + (j) - 1) * 128)
#define SLABP(j) (ws + WS_SLAB + (size_t)((((b & ~3) | ((kq + (j)) & 3)) * 3 + ((j) - 1)) * 4 + wv) * 4096 + lane * 16)
#define STSLAB(j, h) do { const f32x4 a_ = acc[j][2 * (h)], c_ = acc[j][2 * (h) + 1]; \
    st16_loc(SLABP(j) + (h) * 1024, u32x4{cvt2f(a_.x, a_.y), cvt2f(a_.z, a_.w), cvt2f(c_.x, c_.y), cvt2f(c_.z, c_.w)}); } while (0)
  unsigned char* rcv = ws + WS_STG + (size_t)(call & 1) * (4u << 20) + (size_t)(b * 4 + wv) * 4096;
  {
    u32x4 Y[2][4];
    const int ro = lane * 16;
    unsigned cpf;
    if constexpr (MT == 0) CODE_PF(cpf, ".Lz_b0", ".Lz_c0"); else CODE_PF(cpf, ".Lz_b1", ".Lz_c1");
#define XT(tk) ((tk) < 6 ? X[(tk) % 6] : Y[(tk) % 2])
    constexpr int XQ[20][2] = {{3, 1}, {3, 2}, {4, 1}, {5, 1}, {6, 1}, {7, 1}, {4, 2}, {5, 2}, {6, 2}, {7, 2},
                               {4, 3}, {5, 3}, {6, 3}, {7, 3}, {3, 3}, {4, 0}, {5, 0}, {6, 0}, {7, 0}, {3, 0}};
    sfor<64>([&](auto ic) __attribute__((always_inline)) {
      constexpr int n = decltype(ic)::value, tk = n / 16, i = n % 16;
      MF(1 + (i >> 3), i & 7, tk & 1, X[tk][1 + (i >> 3)]);
      if constexpr (tk < 3 && i % 2 == 0) RDF((tk + 1) & 1, ro + (tk + 1) * 8192, i / 2);
      if constexpr (tk == 3) { if constexpr (i < 8) RDF(0, ro + 4 * 8192, i); else RDF(1, ro + 5 * 8192, i - 8); }
      if constexpr (n % 3 == 1 && n / 3 < 20) { constexpr int t2 = XQ[n / 3][0], j = XQ[n / 3][1]; if constexpr (t2 < 6) LDXK(X[t2], 24 + t2, j); else LDXK(Y[t2 - 6], 24 + t2, j); }
      if constexpr (n >= 10 && n < 22) RETH(32768 + wwoff, n - 10);
      if constexpr (n >= 22 && n < 42) RET(U[2], 49152 + wwoff, n - 22);
      SB();
      if constexpr (n == 47) BAR();
    });
    sfor<192>([&](auto ic) __attribute__((always_inline)) {
      constexpr int n = decltype(ic)::value, gq = n / 8, i = n % 8;
      constexpr auto KS = [](int q) { return q < 8 ? 4 + q % 4 : (q + 4) % 8; };
      constexpr int j = gq < 4 ? 1 : gq < 8 ? 2 : gq < 16 ? 3 : 0, tk = KS(gq);
      if constexpr (n == CLR || n == CLR + 4) st16_loc(rcv + (n - CLR) * 256 + lane * 16, u32x4{0u, 0u, 0u, 0u});
      if constexpr (n == 40) { asm volatile("" :: "v"(cpf)); asm volatile("" : A8(1), "+a"(X[3][0])); }
      if constexpr (n == 72) asm volatile("" : A8(2));
      if constexpr (n == 136) asm volatile("" : A8(3));
      if constexpr (tk < 6) MF(j, i, gq & 1, X[tk % 6][j]); else MF(j, i, gq & 1, Y[tk % 2][j]);
      if constexpr (gq + 2 < 24) RDF(gq & 1, ro + KS(gq + 2) * 8192, i);
      if constexpr (n >= 40 && n < 40 + 4 * SSP && (n - 40) % SSP == 0) STSLAB(1, (n - 40) / SSP);
      if constexpr (n >= 72 && n < 72 + 4 * SSP && (n - 72) % SSP == 0) STSLAB(2, (n - 72) / SSP);
      if constexpr (n >= 136 && n < 136 + 4 * SSP && (n - 136) % SSP == 0) STSLAB(3, (n - 136) / SSP);
      if constexpr (n == FL1) { asm volatile("s_waitcnt vmcnt(6)" ::: "memory"); flag_loc(XF(1), call, lane); }
      if constexpr (n == FL2) { asm volatile("s_waitcnt vmcnt(1)" ::: "memory"); flag_loc(XF(2), call, lane); }
      SB();
    });
    DRAIN();
    flag_loc(XF(3), call, lane);
    asm volatile("s_nop 15" : A8(0));
  }
  f32x4 y[8];
  unsigned amax;
  {
    const unsigned char* f = ws + WS_XFLAG + (size_t)((b * 4 + wv) * 4 + (lane & 3)) * 128;
    const __amdgpu_buffer_rsrc_t rs = __builtin_amdgcn_make_buffer_rsrc((void*)(ws + WS_SLAB), 0, 256 * 3 * 4 * 4096, RSRC_W3);
    Waiter wt(tmo, call);
    u32x4 pv[3][4];
    unsigned have = 0;
#pragma unroll
    for (int s = 0; s < 3; ++s) {
      while (!(have >> s & 1u) && !bad) {
        have = (unsigned)__ballot(lane < 3 && ld_loc(f) >= call);
        if (wt.giveup()) bad = true;
      }
      ORDER();
#pragma unroll
      for (int h = 0; h < 4; ++h) pv[s][h] = __builtin_amdgcn_raw_buffer_load_b128(rs, lane * 16 + h * 1024, ((b * 3 + s) * 4 + wv) * 4096, LOC);
      SB();
    }
#pragma unroll
    for (int f2 = 0; f2 < 8; ++f2) y[f2] = acc[0][f2];
#pragma unroll
    for (int s = 0; s < 3; ++s)
#pragma unroll
      for (int h = 0; h < 4; ++h) {
        const u32x4 v = pv[s][h];
        y[2 * h] += f32x4{asf(v.x << 16), asf(v.x & 0xffff0000u), asf(v.y << 16), asf(v.y & 0xffff0000u)};
        y[2 * h + 1] += f32x4{asf(v.z << 16), asf(v.z & 0xffff0000u), asf(v.w << 16), asf(v.w & 0xffff0000u)};
      }
    float am = 0.f;
#pragma unroll
    for (int f2 = 0; f2 < 8; ++f2) am = fmaxf(am, fmaxf(fmaxf(y[f2].x, y[f2].y), fmaxf(y[f2].z, y[f2].w)));
    amax = wave_max(asu(am));
    if (bad) amax = BADMAX;
  }
  LAS unsigned* sm = (LAS unsigned*)(L + LDS_MISC);
  if (lane == 0) sm[wv] = amax;
  BAR();
  if (tid == 0) {
    const unsigned m = max(max(sm[0], sm[1]), max(sm[2], sm[3]));
    __hip_atomic_store(G((u64*)(ws + WS_AMAX) + b), ((u64)call << 32) | m, __ATOMIC_RELAXED, __HIP_MEMORY_SCOPE_AGENT);
  }
  if (tid == 128) __hip_atomic_store(G((unsigned*)(ws + WS_CNT + b * 128)), call, __ATOMIC_RELAXED, __HIP_MEMORY_SCOPE_AGENT);
  const int snd = nsync == NR ? rank * 32 + (ntg & 3) * 8 + MT * 4 + kq : b;
  unsigned char* stg = ws_dest + WS_STG + (size_t)(call & 1) * (4u << 20) + (size_t)(snd * 4 + wv) * 4096;
  f32x4 d0[4], d1[4];
  u32x4 pa;
  unsigned total, tagw;
  u64 mv = __hip_atomic_load(G((const u64*)(rcv + 2048) + lane), __ATOMIC_RELAXED, __HIP_MEMORY_SCOPE_SYSTEM);
  SB();
  {
    unsigned m = 0;
#pragma unroll
    for (int e2 = 0; e2 < 32; ++e2) m |= y[4 * (e2 >> 4) + (e2 & 3)][(e2 >> 2) & 3] > 0.f ? 1u << e2 : 0u;
    const unsigned cnt = __builtin_popcount(m), incl = wave_scan(cnt);
    total = ((unsigned)__builtin_amdgcn_readlane((int)incl, 63) + (RUP - 1)) & ~(unsigned)(RUP - 1);
    __hip_atomic_store(G((u64*)(stg + 2048) + lane), ((u64)call << 32) | m, __ATOMIC_RELAXED, __HIP_MEMORY_SCOPE_SYSTEM);
    unsigned pos = wv * 8704 + (incl - cnt) * 4;
    const unsigned dummy = wv * 8704 + 8192 + lane * 4;
#pragma unroll
    for (int e2 = 0; e2 < 32; ++e2) {
      const float v = y[4 * (e2 >> 4) + (e2 & 3)][(e2 >> 2) & 3];
      *(LAS float*)(L + (v > 0.f ? pos : dummy)) = v;
      pos += v > 0.f ? 4u : 0u;
    }
    asm volatile("" ::: "memory");
    const unsigned tg = (call >> 1) % 3u + 1u;
    tagw = (tg & 1u) << 15 | (tg >> 1) << 31;
    const f32x4 a0 = *(const LAS f32x4*)(L + wv * 8704 + lane * 32), a1 = *(const LAS f32x4*)(L + wv * 8704 + lane * 32 + 16);
    pa = (u32x4{cvt2f(a0.x, a0.y), cvt2f(a0.z, a0.w), cvt2f(a1.x, a1.y), cvt2f(a1.z, a1.w)} & 0x7fff7fffu) | tagw;
#pragma unroll
    for (int q = 0; q < 4; ++q) {
      d0[q] = *(const LAS f32x4*)(L + wv * 8704 + 2048 + lane * 64 + q * 16);
      d1[q] = *(const LAS f32x4*)(L + wv * 8704 + 6144 + (lane & 31) * 64 + q * 16);
    }
  }
  asm volatile("" : "+v"(mv));
  const u64 tpack = __builtin_amdgcn_s_memrealtime();
  const u64 tsend = tpack + (u64)(PCLS);
  bool sent = false;
  if (wv == 0) { st16_sys(stg + 3072 + lane * 16, pa); sent = true; }
  if (wv == 1 && b == 32 * rank) {
    const __amdgpu_buffer_rsrc_t rc = __builtin_amdgcn_make_buffer_rsrc((void*)(ws + WS_AMAX), 0, 2048, RSRC_W3);
    Waiter wt(tmo, call);
    unsigned gm;
    for (;;) {
      const u32x4 a = __builtin_amdgcn_raw_buffer_load_b128(rc, lane * 16, 0, LOC);
      const u32x4 c = __builtin_amdgcn_raw_buffer_load_b128(rc, lane * 16, 1024, LOC);
      gm = max(max(a.x, a.z), max(c.x, c.z));
      if (__all(a.y == call && a.w == call && c.y == call && c.w == call)) break;
      if (wt.giveup()) { gm = BADMAX; break; }
      asm volatile("" ::: "memory");
    }
    gm = wave_max(gm);
    unsigned char* ws_lane = (unsigned char*)(((u64)wl_hi << 32) | wl_lo);
    if (lane < nsync)
      __hip_atomic_store(G((u64*)(ws_lane + WS_GMAX + (call & 1) * 1024) + rank), ((u64)call << 32) | gm, __ATOMIC_RELAXED, __HIP_MEMORY_SCOPE_SYSTEM);
  }
  unsigned sel[8], src8[8], rtotal;
  {
    Waiter wt(tmo, call);
    while (!__all((unsigned)(mv >> 32) == call)) {
      if (!sent && __builtin_amdgcn_s_memrealtime() >= tsend) { st16_sys(stg + 3072 + lane * 16, pa); sent = true; }
      mv = __hip_atomic_load(G((const u64*)(rcv + 2048) + lane), __ATOMIC_RELAXED, __HIP_MEMORY_SCOPE_SYSTEM);
      if (wt.giveup()) { bad = true; break; }
      __builtin_amdgcn_s_sleep(MSL);
    }
    const unsigned m = (unsigned)mv, cnt = __builtin_popcount(m), incl = wave_scan(cnt);
    rtotal = (unsigned)__builtin_amdgcn_readlane((int)incl, 63);
    unsigned pos = incl - cnt;
#pragma unroll
    for (int k = 0; k < 8; ++k) {
      const unsigned nib = (m >> (4 * k)) & 15u;
      const unsigned sb = (nib * 0x00204081u) & 0x01010101u;
      const unsigned rk = sb * 0x01010100u + (pos & 3u) * 0x01010101u;
      sel[k] = (rk & (sb * 0xffu)) | ((sb ^ 0x01010101u) * 0x0cu);
      src8[k] = wv * 8704 + (pos & ~3u);
      pos += __builtin_popcount(nib);
      asm volatile("" : "+v"(sel[k]), "+v"(src8[k]));
    }
    if (wv == 0) {
      const u64* gp = lane == NR ? (const u64*)tmo : (const u64*)(ws + WS_GMAX + (call & 1) * 1024) + (lane < nsync ? lane : 0);
      unsigned gm;
      for (;;) {
        u64 v = (u64)call << 32;
        if (lane <= NR) v = __hip_atomic_load(G(gp), __ATOMIC_RELAXED, __HIP_MEMORY_SCOPE_SYSTEM);
        gm = lane == NR ? ((unsigned)v == call ? BADMAX : 0u) : (unsigned)v;
        if (__all(lane == NR || (unsigned)(v >> 32) == call)) break;
        if (wt.giveup()) { gm = BADMAX; break; }
      }
      gm = wave_max(gm);
      if (lane == 0) { sm[4] = gm; *(volatile LAS unsigned*)(L + LDS_MISC + 20) = 1u; }
    } else {
      while (__builtin_amdgcn_s_memrealtime() < tsend && RFL(*(volatile LAS unsigned*)(L + LDS_MISC + 20)) == 0u) __builtin_amdgcn_s_sleep(1);
    }
    if (!sent) st16_sys(stg + 3072 + lane * 16, pa);
  }
  unsigned char* dst = nsync == NR ? out_own + (size_t)((b >> 5) * M + ((b >> 2) & 1) * 256 + wv * 64 + kq * 16 + r) * NS + ((b >> 3) & 3) * 128 + 16 * g
                                   : out_dest + (size_t)(MT * 256 + wv * 64 + kq * 16 + r) * NS + (ntg & 3) * 128 + 16 * g;
  stg += lane * 16;
  asm volatile("" : "+v"(dst), "+v"(stg));
  BAR();
  const unsigned gbits = sm[4];
  bad = bad || gbits >= BADMAX;
  const float gmax = asf(gbits);
  const float inv_scale = gmax > 0.f ? 448.f / gmax : 0.f;
  const unsigned mark = bad ? 0xffffffffu : 0x80808080u;
  {
    const u32x4 o = quant16(d0, inv_scale) | mark;
    if (512 + lane * 16 < total) st16_sys(stg, o);
    if (total > 1536) {
      const u32x4 o1 = quant16(d1, inv_scale) | mark;
      if (1536 + lane * 16 < total) st16_sys(stg + 1024, o1);
    }
  }
  DRAIN();
  {
    const unsigned rb = rtotal > 512u ? rtotal - 512u : 0u;
    const __amdgpu_buffer_rsrc_t rg = __builtin_amdgcn_make_buffer_rsrc((void*)rcv, 0, (rb + 15u) & ~15u, RSRC_W3);
    const __amdgpu_buffer_rsrc_t ra = __builtin_amdgcn_make_buffer_rsrc((void*)(rcv + 3072), 0, 1024, RSRC_W3);
    Waiter wt(tmo, call);
    u32x4 a, c, h;
    for (;;) {
      h = __builtin_amdgcn_raw_buffer_load_b128(ra, lane * 16, 0, 17);
      a = __builtin_amdgcn_raw_buffer_load_b128(rg, lane * 16, 0, 17);
      c = __builtin_amdgcn_raw_buffer_load_b128(rg, lane * 16, 1024, 17);
      const bool okh = (((h.x ^ tagw) | (h.y ^ tagw) | (h.z ^ tagw) | (h.w ^ tagw)) & 0x80008000u) == 0u;
      const bool oka = lane * 16 >= rb || (a.x & a.y & a.z & a.w & 0x80808080u) == 0x80808080u;
      const bool okc = 1024 + lane * 16 >= rb || (c.x & c.y & c.z & c.w & 0x80808080u) == 0x80808080u;
      if (__all(okh && oka && okc) || bad) break;
      if (wt.giveup()) { bad = true; break; }
      asm volatile("" ::: "memory");
    }
    h &= 0x7fff7fffu;
#define Q2(w, old, hi) __builtin_amdgcn_cvt_pk_fp8_f32(fminf(asf((w) << 16) * inv_scale, 448.f), fminf(asf((w) & 0xffff0000u) * inv_scale, 448.f), old, hi)
    typedef unsigned u32x2 __attribute__((ext_vector_type(2)));
    *(LAS u32x2*)(L + wv * 8704 + lane * 8) = u32x2{(unsigned)Q2(h.y, Q2(h.x, 0, false), true), (unsigned)Q2(h.w, Q2(h.z, 0, false), true)};
    *(LAS u32x4*)(L + wv * 8704 + 512 + lane * 16) = a;
    *(LAS u32x4*)(L + wv * 8704 + 1536 + lane * 16) = c;
    asm volatile("" ::: "memory");
    unsigned o[8];
#pragma unroll
    for (int k = 0; k < 8; ++k) {
      const unsigned lo = *(const LAS unsigned*)(L + src8[k]), hi = *(const LAS unsigned*)(L + src8[k] + 4);
      o[k] = bad ? 0x7f7f7f7fu : __builtin_amdgcn_perm(hi, lo, sel[k]) & 0x7f7f7f7fu;
    }
    st16_sys(dst, u32x4{o[0], o[1], o[2], o[3]});
    st16_sys(dst + 64, u32x4{o[4], o[5], o[6], o[7]});
  }
  if constexpr (MT == 0) asm volatile(".Lz_c0:"); else asm volatile(".Lz_c1:");
}

__global__ void __launch_bounds__(256, 1) fused(Params p) {
  const int tid = threadIdx.x, b = blockIdx.x;
  unsigned call;
  asm volatile("s_mov_b64 exec, 1\n\tglobal_load_dword %0, %1, %2 sc1\n\ts_mov_b64 exec, -1" : "=v"(call) : "v"((int)WS_CNT + b * 128), "s"(p.wsl) : "memory");
  const __amdgpu_buffer_rsrc_t rxi = __builtin_amdgcn_make_buffer_rsrc((void*)p.x, 0, M * K * 4, RSRC_W3);
  const int vo = (((tid >> 6) * 32 + ((tid & 63) >> 3)) * K + (tid & 7) * 4) * 4;
  const int so = (((b >> 2) & 3) * 128 * K + (b & 3) * 1024 + (b >> 4) * 64) * 4;
  u32x4 t[8];
#pragma unroll
  for (int i = 0; i < 4; ++i) {
    t[2 * i] = __builtin_amdgcn_raw_buffer_load_b128(rxi, vo, so + i * 8 * (K * 4), XNT);
    t[2 * i + 1] = __builtin_amdgcn_raw_buffer_load_b128(rxi, vo + 128, so + i * 8 * (K * 4), XNT);
  }
  SB();
  if (b & 8) body<1>(p, t, call); else body<0>(p, t, call);
}

#define HIP_CHECK(expr) do { hipError_t _e = (expr); if (_e != hipSuccess) { \
    fprintf(stderr, "HIP error %s at %s:%d\n", hipGetErrorString(_e), __FILE__, __LINE__); \
    return; } } while (0)

extern "C" void kernel_launch(int rank, int nranks, ncclComm_t comm, hipStream_t stream,
                              void* const* d_in, const int* in_sizes, int n_in,
                              void* d_out, int out_size, const int* in_dtypes, int out_dtype,
                              void* const* d_out_all, void* const* d_ws_all, size_t ws_bytes) {
  if (nranks != NR || (unsigned)rank >= (unsigned)NR || n_in < 2 || in_sizes[0] != M * K || in_sizes[1] != K * N ||
      out_size != NR * M * NS || ws_bytes < WS_NEED) {
    fprintf(stderr, "unexpected problem shape\n");
    return;
  }
  static bool init = false;
  if (!init) {
    HIP_CHECK(hipFuncSetAttribute((const void*)fused, hipFuncAttributeMaxDynamicSharedMemorySize, LDS_BYTES));
    init = true;
  }
  Params p;
  p.x = (const float*)d_in[0];
  p.w = (const float*)d_in[1];
  for (int i = 0; i < NR; ++i) { p.out[i] = (unsigned char*)d_out_all[i]; p.ws[i] = (unsigned char*)d_ws_all[i]; }
  p.rank = rank;
  p.wsl = (unsigned char*)d_ws_all[rank];
  p.outl = (unsigned char*)d_out;
  p.pad = 0;
  hipLaunchKernelGGL(fused, dim3(NWG), dim3(256), LDS_BYTES, stream, p);
  HIP_CHECK(hipGetLastError());
}
```

```cpp
#include <hip/hip_runtime.h>
#include <rccl/rccl.h>
#include <cstdio>
#include <type_traits>
#include <utility>

typedef unsigned u32x4 __attribute__((ext_vector_type(4)));
typedef float f32x4 __attribute__((ext_vector_type(4)));
typedef float f32x2 __attribute__((ext_vector_type(2)));
typedef __bf16 bf16x2 __attribute__((ext_vector_type(2)));
typedef unsigned long long u64;
#define LAS __attribute__((address_space(3)))
#define GAS __attribute__((address_space(1)))
template <class T> __device__ __forceinline__ GAS T* G(T* p) { return (GAS T*)p; }

#define ROT (nt + 8 * nh)
#define QUIET 44
#define RET0 22
#define EA 4
#define SSP 5
#define FL1 104
#define FL2 134
#define CLR 60
#define RUP 64
#define PDLY 50
#define PCLS (wv * PDLY)
#define XNT 2

constexpr int NR = 8, M = 512, K = 4096, N = 4096, NS = N / NR;
constexpr int NWG = 256;
constexpr int RSRC_W3 = 0x00020000;
constexpr int LOC = 16;
constexpr u64 DEADLINE = 50000000ull;
constexpr size_t WS_TMO = 0;
constexpr size_t WS_GMAX = 1024;
constexpr size_t WS_CNT = 4096;
constexpr size_t WS_AMAX = 65536;
constexpr size_t WS_CFLAG = 131072;
constexpr size_t WS_XFLAG = 524288;
constexpr size_t WS_XB = 1u << 20;
constexpr size_t WS_SLAB = 8u << 20;
constexpr size_t WS_STG = 24u << 20;
constexpr size_t WS_NEED = WS_STG + (8u << 20);
constexpr int LDS_MISC = 6 * 16384;
constexpr int LDS_BYTES = LDS_MISC + 256;
constexpr unsigned NA = 384;
constexpr unsigned BADMAX = 0x7f800000u;

struct Params {
  const float* x; const float* w;
  unsigned char* wsl;
  unsigned char* outl;
  int rank; int pad;
  unsigned char* ws[NR];
};

__device__ __forceinline__ float asf(unsigned a) { return __builtin_bit_cast(float, a); }
__device__ __forceinline__ unsigned asu(float a) { return __builtin_bit_cast(unsigned, a); }
__device__ __forceinline__ unsigned cvt2f(float a, float b) {
  f32x2 v = {a, b};
  return __builtin_bit_cast(unsigned, __builtin_convertvector(v, bf16x2));
}
__device__ __forceinline__ unsigned cvt2(unsigned a, unsigned b) { return cvt2f(asf(a), asf(b)); }
#define SB() __builtin_amdgcn_sched_barrier(0)
#define RFL(x) __builtin_amdgcn_readfirstlane(x)
#define BAR() do { __builtin_amdgcn_s_waitcnt(0xC07F); asm volatile("" ::: "memory"); __builtin_amdgcn_s_barrier(); asm volatile("" ::: "memory"); } while (0)
#define DRAIN() asm volatile("s_waitcnt vmcnt(0)" ::: "memory")
#define ORDER() __builtin_amdgcn_fence(__ATOMIC_ACQUIRE, "wavefront")
__device__ __forceinline__ void st16_sys(void* p, u32x4 v) {
  asm volatile("global_store_dwordx4 %0, %1, off sc0 sc1\n\ts_nop 1" :: "v"(p), "v"(v) : "memory");
}
__device__ __forceinline__ void st16_loc(void* p, u32x4 v) {
  asm volatile("global_store_dwordx4 %0, %1, off sc1\n\ts_nop 1" :: "v"(p), "v"(v) : "memory");
}
#define FLINE(pol) asm volatile("s_mov_b64 exec, 0xff\n\tglobal_store_dwordx4 %0, %1, off " pol "\n\ts_mov_b64 exec, -1\n\ts_nop 0" \
                                :: "v"((unsigned char*)p + (lane & 7) * 16), "v"(u32x4{v, v, v, v}) : "memory")
__device__ __forceinline__ void flag_loc(void* p, unsigned v, int lane) { FLINE("sc1"); }
__device__ __forceinline__ unsigned ld_loc(const void* p) { return __hip_atomic_load(G((const unsigned*)p), __ATOMIC_RELAXED, __HIP_MEMORY_SCOPE_AGENT); }
template <int CTRL, int ROWMASK> __device__ __forceinline__ unsigned dpp_max(unsigned v) {
  return max(v, (unsigned)__builtin_amdgcn_update_dpp(0, (int)v, CTRL, ROWMASK, 0xf, false));
}
__device__ __forceinline__ unsigned wave_max(unsigned v) {
  v = dpp_max<0x111, 0xf>(v); v = dpp_max<0x112, 0xf>(v); v = dpp_max<0x114, 0xf>(v); v = dpp_max<0x118, 0xf>(v);
  v = dpp_max<0x142, 0xa>(v); v = dpp_max<0x143, 0xc>(v);
  return (unsigned)__builtin_amdgcn_readlane((int)v, 63);
}
template <int CTRL, int ROWMASK> __device__ __forceinline__ unsigned dpp_add(unsigned v) {
  return v + (unsigned)__builtin_amdgcn_update_dpp(0, (int)v, CTRL, ROWMASK, 0xf, false);
}
__device__ __forceinline__ unsigned wave_scan(unsigned v) {
  v = dpp_add<0x111, 0xf>(v); v = dpp_add<0x112, 0xf>(v); v = dpp_add<0x114, 0xf>(v); v = dpp_add<0x118, 0xf>(v);
  v = dpp_add<0x142, 0xa>(v); v = dpp_add<0x143, 0xc>(v);
  return v;
}
__device__ __forceinline__ u32x4 quant16(const f32x4 (&d)[4], float inv) {
  u32x4 o;
#pragma unroll
  for (int q = 0; q < 4; ++q) {
    int pk = __builtin_amdgcn_cvt_pk_fp8_f32(fminf(d[q].x * inv, 448.f), fminf(d[q].y * inv, 448.f), 0, false);
    o[q] = (unsigned)__builtin_amdgcn_cvt_pk_fp8_f32(fminf(d[q].z * inv, 448.f), fminf(d[q].w * inv, 448.f), pk, true);
  }
  return o;
}
template <int I> using IC = std::integral_constant<int, I>;
template <class F, int... Is> __device__ __forceinline__ void sfor_impl(F&& f, std::integer_sequence<int, Is...>) { (f(IC<Is>{}), ...); }
template <int NN, class F> __device__ __forceinline__ void sfor(F&& f) { sfor_impl(f, std::make_integer_sequence<int, NN>{}); }
constexpr int late_load(int i) { for (int k = 0; k < 18; ++k) if (i == QUIET + (k * (96 - QUIET)) / 18) return k; return -1; }
struct Waiter {
  unsigned* tmo; unsigned call; u64 t0; unsigned n;
  __device__ __forceinline__ Waiter(unsigned* t, unsigned c) : tmo(t), call(c), t0(__builtin_amdgcn_s_memrealtime()), n(0) {}
  __device__ __forceinline__ bool giveup() {
    if ((++n & 63u) == 0u) {
      if (RFL(ld_loc(tmo)) == call || __builtin_amdgcn_s_memrealtime() - t0 > DEADLINE) {
        __hip_atomic_store(G(tmo), call, __ATOMIC_RELAXED, __HIP_MEMORY_SCOPE_AGENT);
        return true;
      }
    }
    return false;
  }
};
#define CODE_PF(dst, beg, end) do { u64 pc_; unsigned len_; \
    asm volatile("s_getpc_b64 %0\n" beg ":\n\ts_mov_b32 %1, " end "-" beg : "=s"(pc_), "=s"(len_)); \
    dst = __builtin_amdgcn_raw_buffer_load_b32(__builtin_amdgcn_make_buffer_rsrc((void*)pc_, 0, len_ < 32768u ? len_ : 0u, RSRC_W3), (lane * 4 + wv) * 64, 0, 0); SB(); } while (0)

template <int MT> __device__ __forceinline__ void body(const Params& p, u32x4 (&t)[8], unsigned call) {
  extern __shared__ __attribute__((aligned(16))) unsigned char smem[];
  const int tid = threadIdx.x, lane = tid & 63, wv = RFL(tid >> 6);
  const int b = blockIdx.x;
  unsigned char* ws = (unsigned char*)(((u64)RFL((unsigned)((u64)p.wsl >> 32)) << 32) | (unsigned)RFL((unsigned)(u64)p.wsl));
  unsigned cpf0;
  if constexpr (MT == 0) CODE_PF(cpf0, ".Lz_a0", ".Lz_b0"); else CODE_PF(cpf0, ".Lz_a1", ".Lz_b1");
  const int kq = b & 3, nh = (b >> 2) & 1, nt = b >> 4, ntg = (nt >> 1) * 4 + nh * 2 + (nt & 1);
  const int r = lane & 15, g = lane >> 4;
  const int dest = ntg >> 2;
  unsigned char* ws_dest = p.ws[dest];
  unsigned char* out_own = p.outl;
  int rank = p.rank;
  u64 wsp[NR];
#pragma unroll
  for (int i = 0; i < NR; ++i) wsp[i] = (u64)p.ws[i];
  LAS unsigned char* L = (LAS unsigned char*)smem;
  unsigned* tmo = (unsigned*)(ws + WS_TMO);
  const __amdgpu_buffer_rsrc_t rw = __builtin_amdgcn_make_buffer_rsrc((void*)p.w, 0, K * N * 4, RSRC_W3);
  const __amdgpu_buffer_rsrc_t rx = __builtin_amdgcn_make_buffer_rsrc((void*)(ws + WS_XB), 0, M * K * 2, RSRC_W3);
  const int wvoff = (kq * 1024 + 16 * (wv >> 1) + 4 * g) * (N * 4) + (ntg * 128 + (wv & 1) * 64 + 4 * r) * 4;
  const int xvoff = (kq * 32) * 32768 + (MT * 16 + wv * 4) * 1024 + lane * 16;
  const int wwoff = wv * 4096 + lane * 16;
  int mbo[4];
#pragma unroll
  for (int j = 0; j < 4; ++j) mbo[j] = ((j + kq) & 3) * 1024;
  const int rot16 = (ROT) & 15;
#define CH(s) (((s) + rot16) & 15)
#define LDWJ(dst, c, j) dst[j] = __builtin_amdgcn_raw_buffer_load_b128(rw, wvoff, (c) * (64 * N * 4) + (((j) & 3) + 32 * ((j) >> 2)) * (N * 4), 0)
#define LDXK(dst, tk, j) dst[j] = __builtin_amdgcn_raw_buffer_load_b128(rx, xvoff, mbo[j] + (CH((tk) >> 1) * 2 + ((tk) & 1)) * 32768, LOC)
#define CV(src, e, q) tf[q] = cvt2(src[2 * (q)][e], src[2 * (q) + 1][e])
#define RET(src, off, n) do { if ((n) % 5 < 4) CV(src, (n) / 5, (n) % 5 & 3); else *(LAS u32x4*)(L + (off) + ((n) / 5) * 1024) = tf; } while (0)
#define RETALL(src, off) do { _Pragma("unroll") for (int n_ = 0; n_ < 20; ++n_) RET(src, off, n_); } while (0)
#define CVH(n) H[n] = cvt2(U[1][2 * ((n) & 1)][(n) >> 1], U[1][2 * ((n) & 1) + 1][(n) >> 1])
#define RETH(off, n) do { if ((n) % 3 < 2) CV(U[1], (n) / 3, 2 + ((n) % 3 & 1)); \
    else *(LAS u32x4*)(L + (off) + ((n) / 3) * 1024) = u32x4{H[2 * ((n) / 3)], H[2 * ((n) / 3) + 1], tf[2], tf[3]}; } while (0)
#define RDF(h, off, f) fw[h][f] = *(const LAS u32x4*)(L + (off) + (f) * 1024)
#define MF(j, f, h, xb) asm volatile("v_mfma_f32_16x16x32_bf16 %0, %1, %2, %0" : "+a"(acc[j][f]) : "v"(fw[h][f]), "a"(xb))
#define A8(j) "+a"(acc[j][0]), "+a"(acc[j][1]), "+a"(acc[j][2]), "+a"(acc[j][3]), "+a"(acc[j][4]), "+a"(acc[j][5]), "+a"(acc[j][6]), "+a"(acc[j][7])
  u32x4 X[6][4], U[3][8], fw[2][8], tf;
  unsigned H[8];
  f32x4 acc[4][8];
  bool bad = false;
  unsigned wl_lo = 0, wl_hi = 0;
  unsigned fall;
  const unsigned char* fp = ws + WS_CFLAG + (size_t)(((kq * 2 + MT) * 8 + wv * 2) * 16 + (lane & 31)) * 128;
  {
    u32x4 wa[8], wb[8];
    const int c0 = CH(MT), c1 = CH(1 - MT), c2 = CH(2), c3 = CH(3);
#define PL(n) do { if ((n) < 8) LDWJ(wa, c0, (n) & 7); else LDWJ(U[1], c2, 2 * MT + ((n) & 1)); } while (0)
#pragma unroll
    for (int n = 0; n < EA; ++n) PL(n);
    SB();
    u32x4 z4 = {0u, 0u, 0u, 0u};
    asm volatile("s_nop 1" : "+v"(z4));
#pragma unroll
    for (int j = 0; j < 4; ++j)
#pragma unroll
      for (int f = 0; f < 8; ++f) asm volatile("v_mfma_f32_16x16x32_bf16 %0, %1, %1, 0" : "=a"(acc[j][f]) : "v"(z4));
    SB();
    const int row0 = MT * 256 + nh * 128 + wv * 32, pc = lane & 7;
    unsigned char* d = ws + WS_XB + (size_t)((kq * 32 + nt * 2 + (pc >> 2)) * 32 + (row0 >> 4)) * 1024 + ((pc & 3) * 16 + (lane >> 3)) * 16;
    u32x4 o[4];
    asm volatile("" : "+v"(t[0]), "+v"(t[1]), "+v"(t[2]), "+v"(t[3]), "+v"(t[4]), "+v"(t[5]), "+v"(t[6]), "+v"(t[7]), "+v"(call));
#pragma unroll
    for (int i = 0; i < 4; ++i) o[i] = u32x4{cvt2(t[2 * i].x, t[2 * i].y), cvt2(t[2 * i].z, t[2 * i].w), cvt2(t[2 * i + 1].x, t[2 * i + 1].y), cvt2(t[2 * i + 1].z, t[2 * i + 1].w)};
    SB();
#pragma unroll
    for (int i = 0; i < 4; ++i) st16_loc(d + (i >> 1) * 1024 + (i & 1) * 128, o[i]);
    call = RFL(call) + 1u;
    if (tid == 0) *(LAS unsigned*)(L + LDS_MISC + 20) = 0u;
    DRAIN();
    flag_loc(ws + WS_CFLAG + (size_t)(((kq * 2 + MT) * 8 + nh * 4 + wv) * 16 + nt) * 128, call, lane);
    SB();
#pragma unroll
    for (int n = EA; n < 10; ++n) PL(n);
    SB();
#pragma unroll
    for (int i = 0; i < NR; ++i)
      asm volatile("v_writelane_b32 %0, %2, %4\n\tv_writelane_b32 %1, %3, %4" : "+v"(wl_lo), "+v"(wl_hi) : "s"((unsigned)wsp[i]), "s"((unsigned)(wsp[i] >> 32)), "n"(i));
    asm volatile("" : "+s"(out_own), "+s"(ws_dest), "+s"(rank));
    RETALL(wa, MT * 16384 + wwoff);
    asm volatile("" :: "v"(cpf0));
    SB();
    const unsigned char* f2 = ws + WS_CFLAG + (size_t)(((kq * 2 + MT) * 8 + wv * 2 + (lane & 1)) * 16 + CH((lane >> 1) & 1)) * 128;
    unsigned f0 = ld_loc(f2);
    SB();
#pragma unroll
    for (int j = 0; j < 8; ++j) LDWJ(wb, c1, j);
    LDWJ(U[1], c2, 2 - 2 * MT); LDWJ(U[1], c2, 3 - 2 * MT);
    SB();
    Waiter wt(tmo, call);
    if (!__all(f0 >= call)) {
      while (!__all(lane >= 4 || ld_loc(f2) >= call)) {
        if (wt.giveup()) { bad = true; break; }
      }
    }
    ORDER();
#pragma unroll
    for (int q = 0; q < 12; ++q) LDXK(X[q >> 2], q >> 2, q & 3);
    SB();
    RETALL(wb, (1 - MT) * 16384 + wwoff);
#pragma unroll
    for (int n = 0; n < 8; ++n) CVH(n);
    SB();
#pragma unroll
    for (int q = 0; q < 12; ++q) { const int n = 12 + (q + 6 * MT) % 12; if (n < 16) LDWJ(U[1], c2, n & 7); else LDWJ(U[2], c3, n & 7); }
    SB();
    fall = ld_loc(fp);
    SB();
  }
  BAR();
#pragma unroll
  for (int f = 0; f < 8; ++f) RDF(0, lane * 16, f);
  SB();
  {
    int rcur = lane * 16, roth = lane * 16 + 49152, wcur = wwoff, woth = wwoff + 49152;
    for (int it = 0; it < 4; ++it) {
      const int cw[3] = {CH(3 * it + 4), CH(3 * it + 5), CH(3 * it + 6)};
      sfor<192>([&](auto ic) __attribute__((always_inline)) {
        constexpr int P = decltype(ic)::value / 96, i = decltype(ic)::value % 96, tt = 3 * P + i / 32, ii = i % 32, k = late_load(i);
        MF(ii >> 3, ii & 7, tt & 1, X[tt][ii >> 3]);
        if constexpr (i < 24 && i % 4 == 0) { constexpr int n = 12 * P + 6 * MT + i / 4; LDWJ(U[n >> 3], cw[n >> 3], n & 7); }
        if constexpr (k >= 12) { constexpr int n = 12 * P + 6 * (1 - MT) + k - 12; LDWJ(U[n >> 3], cw[n >> 3], n & 7); }
        if constexpr (k >= 0 && k < 12) { constexpr int t2 = 3 * P + 3 + k / 4; LDXK(X[t2 % 6], 6 * it + t2, k & 3); }
        if constexpr (ii < 24 && ii % 3 == 1) { if constexpr (tt < 5) RDF((tt + 1) & 1, rcur + (tt + 1) * 8192, ii / 3); else RDF(0, roth, ii / 3); }
        if constexpr (P == 0 && i >= RET0 && i < RET0 + 12) RETH(wcur + 32768, i - RET0);
        if constexpr (P == 0 && i >= RET0 + 12 && i < RET0 + 32) RET(U[2], woth, i - RET0 - 12);
        if constexpr (P == 1 && i >= RET0 && i < RET0 + 20) RET(U[0], woth + 16384, i - RET0);
        if constexpr (P == 1 && i >= RET0 + 20 && i < RET0 + 28) CVH(i - RET0 - 20);
        SB();
        if constexpr (P == 0 && i == QUIET - 1) {
          if (it == 0) {
            asm volatile("" : "+v"(fall));
            if (!__all(fall >= call) && !bad) {
              Waiter wt(tmo, call);
              while (!__all(lane >= 32 || ld_loc(fp) >= call)) {
                if (wt.giveup()) { bad = true; break; }
              }
            }
            ORDER();
          }
        }
        if constexpr (i == 95) { BAR(); }
      });
      const int sw = rcur; rcur = roth; roth = sw;
      const int sw2 = wcur; wcur = woth; woth = sw2;
    }
  }
#define XF(j) (ws + WS_XFLAG + (size_t)((((b & ~3) | ((kq + (j)) & 3)) * 4 + wv) * 4 + (j) - 1) * 128)
#define SLABP(j) (ws + WS_SLAB + (size_t)((((b & ~3) | ((kq + (j)) & 3)) * 3 + ((j) - 1)) * 4 + wv) * 4096 + lane * 16)
#define STSLAB(j, h) do { const f32x4 a_ = acc[j][2 * (h)], c_ = acc[j][2 * (h) + 1]; \
    st16_loc(SLABP(j) + (h) * 1024, u32x4{cvt2f(a_.x, a_.y), cvt2f(a_.z, a_.w), cvt2f(c_.x, c_.y), cvt2f(c_.z, c_.w)}); } while (0)
  unsigned char* rcv = ws + WS_STG + (size_t)(call & 1) * (4u << 20) + (size_t)(b * 4 + wv) * 4096;
  {
    u32x4 Y[2][4];
    const int ro = lane * 16;
    unsigned cpf;
    if constexpr (MT == 0) CODE_PF(cpf, ".Lz_b0", ".Lz_c0"); else CODE_PF(cpf, ".Lz_b1", ".Lz_c1");
    constexpr int XQ[20][2] = {{3, 1}, {3, 2}, {4, 1}, {5, 1}, {6, 1}, {7, 1}, {4, 2}, {5, 2}, {6, 2}, {7, 2},
                               {4, 3}, {5, 3}, {6, 3}, {7, 3}, {3, 3}, {4, 0}, {5, 0}, {6, 0}, {7, 0}, {3, 0}};
    sfor<64>([&](auto ic) __attribute__((always_inline)) {
      constexpr int n = decltype(ic)::value, tk = n / 16, i = n % 16;
      MF(1 + (i >> 3), i & 7, tk & 1, X[tk][1 + (i >> 3)]);
      if constexpr (tk < 3 && i % 2 == 0) RDF((tk + 1) & 1, ro + (tk + 1) * 8192, i / 2);
      if constexpr (tk == 3) { if constexpr (i < 8) RDF(0, ro + 4 * 8192, i); else RDF(1, ro + 5 * 8192, i - 8); }
      if constexpr (n % 3 == 1 && n / 3 < 20) { constexpr int t2 = XQ[n / 3][0], j = XQ[n / 3][1]; if constexpr (t2 < 6) LDXK(X[t2], 24 + t2, j); else LDXK(Y[t2 - 6], 24 + t2, j); }
      if constexpr (n >= 10 && n < 22) RETH(32768 + wwoff, n - 10);
      if constexpr (n >= 22 && n < 42) RET(U[2], 49152 + wwoff, n - 22);
      SB();
      if constexpr (n == 47) BAR();
    });
    sfor<192>([&](auto ic) __attribute__((always_inline)) {
      constexpr int n = decltype(ic)::value, gq = n / 8, i = n % 8;
      constexpr auto KS = [](int q) { return q < 8 ? 4 + q % 4 : (q + 4) % 8; };
      constexpr int j = gq < 4 ? 1 : gq < 8 ? 2 : gq < 16 ? 3 : 0, tk = KS(gq);
      if constexpr (n == CLR || n == CLR + 4) st16_sys(rcv + (n - CLR) * 256 + lane * 16, u32x4{0u, 0u, 0u, 0u});
      if constexpr (n == 40) { asm volatile("" :: "v"(cpf)); asm volatile("" : A8(1), "+a"(X[3][0])); }
      if constexpr (n == 72) asm volatile("" : A8(2));
      if constexpr (n == 136) asm volatile("" : A8(3));
      if constexpr (tk < 6) MF(j, i, gq & 1, X[tk % 6][j]); else MF(j, i, gq & 1, Y[tk % 2][j]);
      if constexpr (gq + 2 < 24) RDF(gq & 1, ro + KS(gq + 2) * 8192, i);
      if constexpr (n >= 40 && n < 40 + 4 * SSP && (n - 40) % SSP == 0) STSLAB(1, (n - 40) / SSP);
      if constexpr (n >= 72 && n < 72 + 4 * SSP && (n - 72) % SSP == 0) STSLAB(2, (n - 72) / SSP);
      if constexpr (n >= 136 && n < 136 + 4 * SSP && (n - 136) % SSP == 0) STSLAB(3, (n - 136) / SSP);
      if constexpr (n == FL1) { asm volatile("s_waitcnt vmcnt(6)" ::: "memory"); flag_loc(XF(1), call, lane); }
      if constexpr (n == FL2) { asm volatile("s_waitcnt vmcnt(1)" ::: "memory"); flag_loc(XF(2), call, lane); }
      SB();
    });
    DRAIN();
    flag_loc(XF(3), call, lane);
    asm volatile("s_nop 15" : A8(0));
  }
  f32x4 y[8];
  unsigned amax;
  {
    const unsigned char* f = ws + WS_XFLAG + (size_t)((b * 4 + wv) * 4 + (lane & 3)) * 128;
    const __amdgpu_buffer_rsrc_t rs = __builtin_amdgcn_make_buffer_rsrc((void*)(ws + WS_SLAB), 0, 256 * 3 * 4 * 4096, RSRC_W3);
    Waiter wt(tmo, call);
    u32x4 pv[3][4];
    unsigned have = 0;
#pragma unroll
    for (int s = 0; s < 3; ++s) {
      while (!(have >> s & 1u) && !bad) {
        have = (unsigned)__ballot(lane < 3 && ld_loc(f) >= call);
        if (wt.giveup()) bad = true;
      }
      ORDER();
#pragma unroll
      for (int h = 0; h < 4; ++h) pv[s][h] = __builtin_amdgcn_raw_buffer_load_b128(rs, lane * 16 + h * 1024, ((b * 3 + s) * 4 + wv) * 4096, LOC);
      SB();
    }
#pragma unroll
    for (int f2 = 0; f2 < 8; ++f2) y[f2] = acc[0][f2];
#pragma unroll
    for (int s = 0; s < 3; ++s)
#pragma unroll
      for (int h = 0; h < 4; ++h) {
        const u32x4 v = pv[s][h];
        y[2 * h] += f32x4{asf(v.x << 16), asf(v.x & 0xffff0000u), asf(v.y << 16), asf(v.y & 0xffff0000u)};
        y[2 * h + 1] += f32x4{asf(v.z << 16), asf(v.z & 0xffff0000u), asf(v.w << 16), asf(v.w & 0xffff0000u)};
      }
    float am = 0.f;
#pragma unroll
    for (int f2 = 0; f2 < 8; ++f2) am = fmaxf(am, fmaxf(fmaxf(y[f2].x, y[f2].y), fmaxf(y[f2].z, y[f2].w)));
    amax = wave_max(asu(am));
    if (bad) amax = BADMAX;
  }
  LAS unsigned* sm = (LAS unsigned*)(L + LDS_MISC);
  if (lane == 0) sm[wv] = amax;
  BAR();
  if (tid == 0) {
    const unsigned m = max(max(sm[0], sm[1]), max(sm[2], sm[3]));
    __hip_atomic_store(G((u64*)(ws + WS_AMAX) + b), ((u64)call << 32) | m, __ATOMIC_RELAXED, __HIP_MEMORY_SCOPE_AGENT);
  }
  if (tid == 128) __hip_atomic_store(G((unsigned*)(ws + WS_CNT + b * 128)), call, __ATOMIC_RELAXED, __HIP_MEMORY_SCOPE_AGENT);
  const int snd = rank * 32 + (ntg & 3) * 8 + MT * 4 + kq;
  unsigned char* stg = ws_dest + WS_STG + (size_t)(call & 1) * (4u << 20) + (size_t)(snd * 4 + wv) * 4096;
  f32x4 d0[4], d1[4];
  u32x4 pa;
  unsigned total;
  const unsigned tagw = (~((call - 1u) >> 1) & 1u) << 31;
  {
    unsigned m = 0;
#pragma unroll
    for (int e2 = 0; e2 < 32; ++e2) m |= y[4 * (e2 >> 4) + (e2 & 3)][(e2 >> 2) & 3] > 0.f ? 1u << e2 : 0u;
    const unsigned cnt = __builtin_popcount(m), incl = wave_scan(cnt);
    total = ((unsigned)__builtin_amdgcn_readlane((int)incl, 63) + (RUP - 1)) & ~(unsigned)(RUP - 1);
    unsigned pos = wv * 8704 + (incl - cnt) * 4;
    const unsigned dummy = wv * 8704 + 8192 + lane * 4;
#pragma unroll
    for (int e2 = 0; e2 < 32; ++e2) {
      const float v = y[4 * (e2 >> 4) + (e2 & 3)][(e2 >> 2) & 3];
      *(LAS float*)(L + (v > 0.f ? pos : dummy)) = v;
      pos += v > 0.f ? 4u : 0u;
    }
    asm volatile("" ::: "memory");
    const LAS f32x2* ap = (const LAS f32x2*)(L + wv * 8704 + lane * 24);
    const f32x2 a0 = ap[0], a1 = ap[1], a2 = ap[2];
    pa = u32x4{m & 0x7fffffffu, (cvt2f(a0.x, a0.y) & 0x7fff7fffu) | (m >> 31) << 15, cvt2f(a1.x, a1.y) & 0x7fff7fffu, cvt2f(a2.x, a2.y) & 0x7fff7fffu} | tagw;
#pragma unroll
    for (int q = 0; q < 4; ++q) {
      d0[q] = *(const LAS f32x4*)(L + wv * 8704 + NA * 4 + lane * 64 + q * 16);
      d1[q] = *(const LAS f32x4*)(L + wv * 8704 + (NA + 1024) * 4 + min(lane, 39) * 64 + q * 16);
    }
  }
  stg += lane * 16;
  const u64 tsend = __builtin_amdgcn_s_memrealtime() + (u64)(PCLS);
  if (wv == 0 || b == 32 * rank) st16_sys(stg + 2048, pa);
  if (wv == 1 && b == 32 * rank) {
    const __amdgpu_buffer_rsrc_t rc = __builtin_amdgcn_make_buffer_rsrc((void*)(ws + WS_AMAX), 0, 2048, RSRC_W3);
    Waiter wt(tmo, call);
    unsigned gm;
    for (;;) {
      const u32x4 a = __builtin_amdgcn_raw_buffer_load_b128(rc, lane * 16, 0, LOC);
      const u32x4 c = __builtin_amdgcn_raw_buffer_load_b128(rc, lane * 16, 1024, LOC);
      gm = max(max(a.x, a.z), max(c.x, c.z));
      if (__all(a.y == call && a.w == call && c.y == call && c.w == call)) break;
      if (wt.giveup()) { gm = BADMAX; break; }
      asm volatile("" ::: "memory");
    }
    gm = wave_max(gm);
    unsigned char* ws_lane = (unsigned char*)(((u64)wl_hi << 32) | wl_lo);
    if (lane < NR)
      __hip_atomic_store(G((u64*)(ws_lane + WS_GMAX + (call & 1) * 1024) + rank), ((u64)call << 32) | gm, __ATOMIC_RELAXED, __HIP_MEMORY_SCOPE_SYSTEM);
  }
  if (wv == 0) {
    const u64* gp = lane == NR ? (const u64*)tmo : (const u64*)(ws + WS_GMAX + (call & 1) * 1024) + (lane < NR ? lane : 0);
    Waiter wt(tmo, call);
    unsigned gm;
    for (;;) {
      u64 v = (u64)call << 32;
      if (lane <= NR) v = __hip_atomic_load(G(gp), __ATOMIC_RELAXED, __HIP_MEMORY_SCOPE_SYSTEM);
      gm = lane == NR ? ((unsigned)v == call ? BADMAX : 0u) : (unsigned)v;
      if (__all(lane == NR || (unsigned)(v >> 32) == call)) break;
      if (wt.giveup()) { gm = BADMAX; break; }
    }
    gm = wave_max(gm);
    if (lane == 0) { sm[4] = gm; *(volatile LAS unsigned*)(L + LDS_MISC + 20) = 1u; }
  } else if (b != 32 * rank) {
    while (__builtin_amdgcn_s_memrealtime() < tsend && RFL(*(volatile LAS unsigned*)(L + LDS_MISC + 20)) == 0u) __builtin_amdgcn_s_sleep(1);
    st16_sys(stg + 2048, pa);
  }
  unsigned char* dst = out_own + (size_t)((b >> 5) * M + ((b >> 2) & 1) * 256 + wv * 64 + kq * 16 + r) * NS + ((b >> 3) & 3) * 128 + 16 * g;
  asm volatile("" : "+v"(dst), "+v"(stg));
  BAR();
  const unsigned gbits = sm[4];
  bad = bad || gbits >= BADMAX;
  const float gmax = asf(gbits);
  float inv_scale = 448.f * __builtin_amdgcn_rcpf(gmax);
  if (__builtin_expect(gbits - 0x10000000u >= 0x60000000u, 0)) inv_scale = gmax > 0.f ? 448.f / gmax : 0.f;
  const unsigned mark = bad ? 0xffffffffu : 0x80808080u;
  u32x4 h;
  {
    const u32x4 o = quant16(d0, inv_scale) | mark;
    const u64 em = __ballot(NA + lane * 16 < total) | 1ull;
    asm volatile("global_load_dwordx4 %0, %4, off offset:2048 sc0 sc1\n\ts_mov_b64 exec, %3\n\tglobal_store_dwordx4 %1, %2, off sc0 sc1\n\t"
                 "s_mov_b64 exec, -1\n\ts_waitcnt vmcnt(1)" : "=&v"(h) : "v"(stg), "v"(o), "s"(em), "v"(rcv + lane * 16) : "memory");
    if (total > NA + 1024) {
      const u32x4 o1 = quant16(d1, inv_scale) | mark;
      if (NA + 1024 + lane * 16 < total) st16_sys(stg + 1024, o1);
    }
  }
  unsigned sel[8], src8[8];
  {
    const __amdgpu_buffer_rsrc_t rg = __builtin_amdgcn_make_buffer_rsrc((void*)rcv, 0, 3072, RSRC_W3);
    Waiter wt(tmo, call);
    if (!__all((((h.x ^ tagw) | (h.y ^ tagw) | (h.z ^ tagw) | (h.w ^ tagw)) >> 31) == 0u) && !bad) {
      for (;;) {
        h = __builtin_amdgcn_raw_buffer_load_b128(rg, lane * 16, 2048, 17);
        if (__all((((h.x ^ tagw) | (h.y ^ tagw) | (h.z ^ tagw) | (h.w ^ tagw)) >> 31) == 0u)) break;
        if (wt.giveup()) { bad = true; break; }
        asm volatile("" ::: "memory");
      }
    }
    const unsigned m = (h.x & 0x7fffffffu) | (h.y >> 15) << 31, cnt = __builtin_popcount(m), incl = wave_scan(cnt);
    const unsigned rtotal = (unsigned)__builtin_amdgcn_readlane((int)incl, 63);
    unsigned pos = incl - cnt;
#pragma unroll
    for (int k = 0; k < 8; ++k) {
      const unsigned nib = (m >> (4 * k)) & 15u;
      const unsigned sb = (nib * 0x00204081u) & 0x01010101u;
      const unsigned rk = sb * 0x01010100u + (pos & 3u) * 0x01010101u;
      sel[k] = (rk & (sb * 0xffu)) | ((sb ^ 0x01010101u) * 0x0cu);
      src8[k] = wv * 8704 + (pos & ~3u);
      pos += __builtin_popcount(nib);
      asm volatile("" : "+v"(sel[k]), "+v"(src8[k]));
    }
#define Q2(w) (unsigned short)__builtin_amdgcn_cvt_pk_fp8_f32(fminf(asf(((w) & 0x7fffu) << 16) * inv_scale, 448.f), fminf(asf((w) & 0x7fff0000u) * inv_scale, 448.f), 0, false)
    LAS unsigned short* qp = (LAS unsigned short*)(L + wv * 8704 + lane * 6);
    qp[0] = Q2(h.y); qp[1] = Q2(h.z); qp[2] = Q2(h.w);
    const int oa = NA + lane * 16 < rtotal ? lane * 16 : 4096, oc = NA + 1024 + lane * 16 < rtotal ? lane * 16 : 4096;
    DRAIN();
    u32x4 a, c;
    for (;;) {
      a = __builtin_amdgcn_raw_buffer_load_b128(rg, oa, 0, 17);
      c = __builtin_amdgcn_raw_buffer_load_b128(rg, oc, 1024, 17);
      const bool oka = oa == 4096 || (a.x & a.y & a.z & a.w & 0x80808080u) == 0x80808080u;
      const bool okc = oc == 4096 || (c.x & c.y & c.z & c.w & 0x80808080u) == 0x80808080u;
      if (__all(oka && okc) || bad) break;
      if (wt.giveup()) { bad = true; break; }
      asm volatile("" ::: "memory");
    }
    *(LAS u32x4*)(L + wv * 8704 + NA + lane * 16) = a;
    *(LAS u32x4*)(L + wv * 8704 + NA + 1024 + lane * 16) = c;
    asm volatile("" ::: "memory");
    unsigned o[8];
#pragma unroll
    for (int k = 0; k < 8; ++k) {
      const unsigned lo = *(const LAS unsigned*)(L + src8[k]), hi = *(const LAS unsigned*)(L + src8[k] + 4);
      o[k] = bad ? 0x7f7f7f7fu : __builtin_amdgcn_perm(hi, lo, sel[k]) & 0x7f7f7f7fu;
    }
    st16_sys(dst, u32x4{o[0], o[1], o[2], o[3]});
    st16_sys(dst + 64, u32x4{o[4], o[5], o[6], o[7]});
  }
  if constexpr (MT == 0) asm volatile(".Lz_c0:"); else asm volatile(".Lz_c1:");
}

__global__ void __launch_bounds__(256, 1) fused(Params p) {
  const int tid = threadIdx.x, b = blockIdx.x;
  unsigned call = ld_loc(p.wsl + WS_CNT + b * 128);
  const __amdgpu_buffer_rsrc_t rxi = __builtin_amdgcn_make_buffer_rsrc((void*)p.x, 0, M * K * 4, RSRC_W3);
  const int vo = (((tid >> 6) * 32 + ((tid & 63) >> 3)) * K + (tid & 7) * 4) * 4;
  const int so = (((b >> 2) & 3) * 128 * K + (b & 3) * 1024 + (b >> 4) * 64) * 4;
  u32x4 t[8];
#pragma unroll
  for (int i = 0; i < 4; ++i) {
    t[2 * i] = __builtin_amdgcn_raw_buffer_load_b128(rxi, vo, so + i * 8 * (K * 4), XNT);
    t[2 * i + 1] = __builtin_amdgcn_raw_buffer_load_b128(rxi, vo + 128, so + i * 8 * (K * 4), XNT);
  }
  SB();
  if (b & 8) body<1>(p, t, call); else body<0>(p, t, call);
}

#define HIP_CHECK(expr) do { hipError_t _e = (expr); if (_e != hipSuccess) { \
    fprintf(stderr, "HIP error %s at %s:%d\n", hipGetErrorString(_e), __FILE__, __LINE__); \
    return; } } while (0)

extern "C" void kernel_launch(int rank, int nranks, ncclComm_t comm, hipStream_t stream,
                              void* const* d_in, const int* in_sizes, int n_in,
                              void* d_out, int out_size, const int* in_dtypes, int out_dtype,
                              void* const* d_out_all, void* const* d_ws_all, size_t ws_bytes) {
  if (nranks != NR || (unsigned)rank >= (unsigned)NR || n_in < 2 || in_sizes[0] != M * K || in_sizes[1] != K * N ||
      out_size != NR * M * NS || ws_bytes < WS_NEED) {
    fprintf(stderr, "unexpected problem shape\n");
    return;
  }
  static bool init = false;
  if (!init) {
    HIP_CHECK(hipFuncSetAttribute((const void*)fused, hipFuncAttributeMaxDynamicSharedMemorySize, LDS_BYTES));
    init = true;
  }
  Params p;
  p.x = (const float*)d_in[0];
  p.w = (const float*)d_in[1];
  for (int i = 0; i < NR; ++i) p.ws[i] = (unsigned char*)d_ws_all[i];
  p.rank = rank;
  p.wsl = (unsigned char*)d_ws_all[rank];
  p.outl = (unsigned char*)d_out;
  p.pad = 0;
  hipLaunchKernelGGL(fused, dim3(NWG), dim3(256), LDS_BYTES, stream, p);
  HIP_CHECK(hipGetLastError());
}
```
